```python
import numpy as np

kernel_name = "gpu_gemm_ar_m4096_k4096_n4096_e5m2e5m2_dqsilu"


ABSMAX_RTOL = 0.05

COMPUTE_DTYPE = "float32"
EPILOGUE = "dequant_silu"
IN_QUANT = None
OUT_QUANT = None
IN_FP8 = "e5m2e5m2"
IN_DTYPES = {"x": "e5m2", "w_mat": "e5m2",
             "scale_x": "float32", "scale_w": "float32"}
WORLD_SIZE = 8

def _snap_fp8(x, mant_bits, max_abs, min_normal):
    a = np.abs(x.astype(np.float64))
    _m, _e = np.frexp(a)
    q = np.where(a >= min_normal, np.ldexp(1.0, _e - 1 - mant_bits),
                 min_normal * 2.0 ** -mant_bits)
    snapped = np.minimum(np.round(a / q) * q, max_abs)
    return (np.sign(x) * snapped).astype(np.float32)

def setup_inputs():

    x = _snap_fp8(np.random.randn(4096, 4096).astype(np.float32), 2, 57344.0, 2.0 ** -14)
    w_mat = _snap_fp8(np.random.randn(4096, 4096).astype(np.float32), 2, 57344.0, 2.0 ** -14)
    scale_x = (0.02 * (1.0 + np.random.rand(1))).astype(np.float32)
    scale_w = (0.02 * (1.0 + np.random.rand(1))).astype(np.float32)
    return {"x": x, "w_mat": w_mat, "scale_x": scale_x, "scale_w": scale_w}

def reference(x, w_mat, scale_x, scale_w):
    acc = (x @ w_mat).astype(np.float32)
    y = (acc * (scale_x * scale_w)).astype(np.float32)
    return (y / (1.0 + np.exp(-np.clip(y, -60.0, 60.0)))).astype(np.float32)
```

```cpp
#include <hip/hip_runtime.h>
#include <rccl/rccl.h>
#include <cstdio>
#include <cstdint>

#define HIP_CHECK(expr) do { hipError_t _e = (expr); if (_e != hipSuccess) { \
    fprintf(stderr, "HIP error %s at %s:%d\n", hipGetErrorString(_e), __FILE__, __LINE__); \
    return; } } while (0)

typedef int v8i __attribute__((ext_vector_type(8)));
typedef int v4i __attribute__((ext_vector_type(4)));
typedef unsigned v4u __attribute__((ext_vector_type(4)));
typedef float v16f __attribute__((ext_vector_type(16)));
typedef float v4f __attribute__((ext_vector_type(4)));
typedef __attribute__((address_space(1))) unsigned gu32;

namespace {
constexpr int MDIM = 4096, NDIM = 4096, KSH = 512, NR = 8;
constexpr int NSL = 16;
constexpr unsigned PIECE = 8192;
constexpr unsigned OPER_BYTES = 16 * PIECE;
constexpr unsigned SLICE_BYTES = 2 * OPER_BYTES;
constexpr unsigned SRC_BYTES = NSL * SLICE_BYTES;
constexpr size_t PAR_BYTES = (size_t)NR * SRC_BYTES;
constexpr size_t FLAG_OFF = 2 * PAR_BYTES;
constexpr size_t CTR_GEMM_OFF = FLAG_OFF + NR * 128;
constexpr size_t CTR_COMM_OFF = CTR_GEMM_OFF + 128;
constexpr size_t TMO_OFF = CTR_COMM_OFF + 128;
constexpr size_t WS_NEEDED = TMO_OFF + 128;
#define TS(slot, idx) do {} while (0)
constexpr int NCW = 64;
constexpr int NGEMM = 256;
constexpr int NSTAGE = 64;
constexpr int RING = 4;
constexpr unsigned STAGE_BYTES = 32768;
constexpr unsigned LDS_BYTES = RING * STAGE_BYTES;
constexpr unsigned long long DEADLINE_TICKS = 50000000ull;

struct Args {
  const uint8_t* x;
  const uint8_t* w;
  const float* sx;
  const float* sw;
  float* out;
  char* ws[NR];
  int rank;
  int role;
};

__device__ __forceinline__ void entry_of(int n, int rank, int& src, int& q) {
  if (n < NSL) { src = rank; q = n; }
  else { int m = n - NSL; q = m / 7; src = (rank + 1 + (m - q * 7)) & 7; }
}

__device__ __forceinline__ unsigned ld_sys(const char* p) {
  return __hip_atomic_load((gu32*)p, __ATOMIC_RELAXED, __HIP_MEMORY_SCOPE_SYSTEM);
}
__device__ __forceinline__ unsigned ld_agent(const char* p) {
  return __hip_atomic_load((gu32*)p, __ATOMIC_RELAXED, __HIP_MEMORY_SCOPE_AGENT);
}

template <int ND>
__device__ __forceinline__ void comm_body(const Args& a, int cw, unsigned par) {
  const int lane = threadIdx.x;
  __amdgpu_buffer_rsrc_t rs[ND];
#pragma unroll
  for (int k = 0; k < ND; ++k) {
    int p = ND == 1 ? a.rank : (a.rank + 1 + (k + cw) % 7) & 7;
    char* base = a.ws[p] + par * PAR_BYTES + (size_t)a.rank * SRC_BYTES;
    rs[k] = __builtin_amdgcn_make_buffer_rsrc(base, 0, SRC_BYTES, 0x00020000);
  }
  char* flagp = a.ws[ND == 1 ? a.rank : (a.rank + 1 + lane) & 7] + FLAG_OFF + a.rank * 128;
  if (cw < 32) {
    const uint8_t* src = a.x + (size_t)(cw * 128 + (lane >> 1)) * KSH + (lane & 1) * 16;
    const unsigned doff = OPER_BYTES + (cw * 128 + (lane >> 1)) * 32 + (lane & 1) * 16;
    v4u cur[4], nxt[4];
#pragma unroll
    for (int u = 0; u < 4; ++u) cur[u] = *(const v4u*)(src + (size_t)u * 32 * KSH);
    for (int q = 0; q < NSL; ++q) {
      if (q + 1 < NSL) {
#pragma unroll
        for (int u = 0; u < 4; ++u) nxt[u] = *(const v4u*)(src + (size_t)u * 32 * KSH + (q + 1) * 32);
      }
#pragma unroll
      for (int u = 0; u < 4; ++u) {
#pragma unroll
        for (int k = 0; k < ND; ++k)
          __builtin_amdgcn_raw_buffer_store_b128(cur[u], rs[k], doff + u * 1024, q * SLICE_BYTES, 17);
        if (ND > 1) asm volatile("s_waitcnt vmcnt(7)" ::: "memory");
      }
      asm volatile("s_waitcnt vmcnt(0)" ::: "memory");
      if (lane < ND) __hip_atomic_fetch_add((gu32*)(flagp + q * 4), 1u, __ATOMIC_RELAXED, __HIP_MEMORY_SCOPE_SYSTEM);
      if (ND > 1) TS(cw, 1 + q);
#pragma unroll
      for (int u = 0; u < 4; ++u) cur[u] = nxt[u];
    }
  } else {
    const int cb = cw - 32, blk = cb >> 1, h = cb & 1;
    const uint8_t* src = a.w + (size_t)(h * 16) * NDIM + blk * 256 + lane * 4;
    const unsigned doff = blk * PIECE + h * 4096 + lane * 16;
    unsigned d[16];
#pragma unroll
    for (int i = 0; i < 16; ++i) d[i] = *(const unsigned*)(src + (size_t)i * NDIM);
    for (int q = 0; q < NSL; ++q) {
      v4u e[4];
#pragma unroll
      for (int g = 0; g < 4; ++g) {
        unsigned t0 = __builtin_amdgcn_perm(d[4 * g + 1], d[4 * g + 0], 0x05010400u);
        unsigned t1 = __builtin_amdgcn_perm(d[4 * g + 1], d[4 * g + 0], 0x07030602u);
        unsigned t2 = __builtin_amdgcn_perm(d[4 * g + 3], d[4 * g + 2], 0x05010400u);
        unsigned t3 = __builtin_amdgcn_perm(d[4 * g + 3], d[4 * g + 2], 0x07030602u);
        e[0][g] = __builtin_amdgcn_perm(t2, t0, 0x05040100u);
        e[1][g] = __builtin_amdgcn_perm(t2, t0, 0x07060302u);
        e[2][g] = __builtin_amdgcn_perm(t3, t1, 0x05040100u);
        e[3][g] = __builtin_amdgcn_perm(t3, t1, 0x07060302u);
      }
      if (q + 1 < NSL) {
#pragma unroll
        for (int i = 0; i < 16; ++i) d[i] = *(const unsigned*)(src + (size_t)((q + 1) * 32 + i) * NDIM);
      }
#pragma unroll
      for (int j = 0; j < 4; ++j) {
#pragma unroll
        for (int k = 0; k < ND; ++k)
          __builtin_amdgcn_raw_buffer_store_b128(e[j], rs[k], doff + j * 1024, q * SLICE_BYTES, 17);
        if (ND > 1) asm volatile("s_waitcnt vmcnt(7)" ::: "memory");
      }
      asm volatile("s_waitcnt vmcnt(0)" ::: "memory");
      if (lane < ND) __hip_atomic_fetch_add((gu32*)(flagp + q * 4), 1u, __ATOMIC_RELAXED, __HIP_MEMORY_SCOPE_SYSTEM);
      if (ND > 1) TS(cw, 1 + q);
    }
  }
}

__device__ __forceinline__ void comm_role(const Args& a) {
  char* myws = a.ws[a.rank];
  const unsigned call = ld_agent(myws + CTR_COMM_OFF) / (2 * NCW) + 1;
  const unsigned par = call & 1;
  if (blockIdx.x < NCW) { TS(blockIdx.x, 0); comm_body<7>(a, blockIdx.x, par); }
  else comm_body<1>(a, blockIdx.x - NCW, par);
  if (threadIdx.x == 0) __hip_atomic_fetch_add((gu32*)(myws + CTR_COMM_OFF), 1u, __ATOMIC_RELAXED, __HIP_MEMORY_SCOPE_AGENT);
}

__device__ __forceinline__ void gemm_role(const Args& a, char* smem) {
  const int tid = threadIdx.x;
  const int lane = tid & 63;
  const int wv = __builtin_amdgcn_readfirstlane(tid >> 6);
  const int rank = a.rank;
  char* myws = a.ws[rank];
  const unsigned call = __builtin_amdgcn_readfirstlane(ld_agent(myws + CTR_GEMM_OFF)) / NGEMM + 1;
  const unsigned par = call & 1;
  const unsigned target = call * NCW;
  if (wv == 1) TS(64 + blockIdx.x, 0);
  if (wv == 0) __builtin_amdgcn_fence(__ATOMIC_ACQUIRE, "");

  const int xcd = blockIdx.x & 7, ii = blockIdx.x >> 3;
  const int bn = (xcd >> 1) * 4 + (ii >> 3);
  const int bm = (xcd & 1) * 8 + (ii & 7);
  const int wn = wv >> 2, wm = wv & 3;

  const int ld_op = wv >> 2;
  const int ld_sl = (wv >> 1) & 1;
  const int ld_rg = (wv & 1) * 4;
  const unsigned hh = (lane & 1) ^ (lane >> 5);
  unsigned voff;
  if (ld_op) voff = OPER_BYTES + bm * PIECE + (ld_rg * 32 + (lane >> 1)) * 32 + hh * 16;
  else       voff = bn * PIECE + hh * 4096 + ((lane >> 1) & 3) * 1024 + (ld_rg * 8 + (lane >> 3)) * 16;
  const unsigned gstep = ld_op ? 1024u : 128u;
  const char* stg = myws + par * PAR_BYTES;

  auto issue = [&](int s) {
    int src, q;
    entry_of(2 * s + ld_sl, rank, src, q);
    const char* g = stg + (size_t)src * SRC_BYTES + (unsigned)q * SLICE_BYTES;
    char* l = smem + (s & (RING - 1)) * STAGE_BYTES + wv * 4096;
#pragma unroll
    for (int i = 0; i < 4; ++i)
      __builtin_amdgcn_global_load_lds((const __attribute__((address_space(1))) void*)(g + voff + i * gstep),
                                       (__attribute__((address_space(3))) void*)(l + i * 1024), 16, 0, 0);
  };

  int nready = 0;
  bool failed = false;
  auto ensure = [&](int s) {
    const int need = 2 * s + 2;
    if (nready >= need) return;
    unsigned long long t0 = __builtin_amdgcn_s_memrealtime();
    for (unsigned it = 1;; ++it) {
      int n = nready + lane;
      bool ok = true;
      if (lane < 4 && n < 2 * NSTAGE) {
        int src, q;
        entry_of(n, rank, src, q);
        ok = ld_sys(myws + FLAG_OFF + src * 128 + q * 4) >= target;
      }
      unsigned long long bal = __ballot(!ok);
      int lead = bal ? __builtin_ctzll(bal) : 64;
      if (lead > 4) lead = 4;
      nready = __builtin_amdgcn_readfirstlane(nready + lead);
      if (nready >= need) break;
      if ((it & 63) == 0 &&
          (ld_agent(myws + TMO_OFF) == call || __builtin_amdgcn_s_memrealtime() - t0 > DEADLINE_TICKS)) {
        if (lane == 0) __hip_atomic_store((gu32*)(myws + TMO_OFF), call, __ATOMIC_RELAXED, __HIP_MEMORY_SCOPE_AGENT);
        nready = 2 * NSTAGE;
        break;
      }
      __builtin_amdgcn_s_sleep(12);
    }
  };

  const int r = lane & 31, kg = lane >> 5;
  const unsigned fa0 = kg * 8192 + (wn * 128 + r) * 32 + ((r >> 4) & 1) * 16;
  const unsigned fb0 = 16384 + kg * 8192 + (wm * 64 + r) * 32 + ((r >> 4) & 1) * 16;

  v16f acc[4][2];
#pragma unroll
  for (int i = 0; i < 4; ++i)
#pragma unroll
    for (int j = 0; j < 2; ++j)
#pragma unroll
      for (int e = 0; e < 16; ++e) acc[i][j][e] = 0.f;

  if (wv == 0) ensure(2);
  __builtin_amdgcn_s_barrier();
  issue(0); issue(1); issue(2);
  if (wv == 0) ensure(3);
  asm volatile("s_waitcnt vmcnt(8)" ::: "memory");
  __builtin_amdgcn_s_barrier();

  for (int s = 0; s < NSTAGE; ++s) {
    if (s + 3 < NSTAGE) issue(s + 3);
    const char* sb = smem + (s & (RING - 1)) * STAGE_BYTES;
    v8i bf[2];
#pragma unroll
    for (int j = 0; j < 2; ++j) {
      v4i lo = *(const v4i*)(sb + fb0 + j * 1024);
      v4i hi = *(const v4i*)(sb + (fb0 ^ 16) + j * 1024);
      bf[j] = __builtin_shufflevector(lo, hi, 0, 1, 2, 3, 4, 5, 6, 7);
    }
#pragma unroll
    for (int i = 0; i < 4; ++i) {
      v4i lo = *(const v4i*)(sb + fa0 + i * 1024);
      v4i hi = *(const v4i*)(sb + (fa0 ^ 16) + i * 1024);
      v8i af = __builtin_shufflevector(lo, hi, 0, 1, 2, 3, 4, 5, 6, 7);
#pragma unroll
      for (int j = 0; j < 2; ++j)
        acc[i][j] = __builtin_amdgcn_mfma_scale_f32_32x32x64_f8f6f4(af, bf[j], acc[i][j], 1, 1, 0, 0, 0, 0);
    }
    if (wv == 0 && s + 4 < NSTAGE) ensure(s + 4);
    if (s + 3 < NSTAGE) asm volatile("s_waitcnt vmcnt(8) lgkmcnt(0)" ::: "memory");
    else if (s + 2 < NSTAGE) asm volatile("s_waitcnt vmcnt(4) lgkmcnt(0)" ::: "memory");
    else asm volatile("s_waitcnt vmcnt(0) lgkmcnt(0)" ::: "memory");
    __builtin_amdgcn_s_barrier();
  }

  if (wv == 1) TS(64 + blockIdx.x, 64);
  const bool bad = ld_agent(myws + TMO_OFF) == call;
  const float sc = a.sx[0] * a.sw[0];
  float* op = a.out + (size_t)(bm * 256 + wm * 64 + r) * NDIM + bn * 256 + wn * 128 + kg * 4;
#pragma unroll
  for (int j = 0; j < 2; ++j)
#pragma unroll
    for (int i = 0; i < 4; ++i)
#pragma unroll
      for (int g = 0; g < 4; ++g) {
        v4f o;
#pragma unroll
        for (int e = 0; e < 4; ++e) {
          float y = acc[i][j][g * 4 + e] * sc;
          float c = fminf(fmaxf(y, -60.f), 60.f);
          float v = y / (1.0f + __expf(-c));
          o[e] = bad ? __builtin_nanf("") : v;
        }
        *(v4f*)(op + (size_t)j * 32 * NDIM + i * 32 + g * 8) = o;
      }
  if (wv == 1) { asm volatile("s_waitcnt vmcnt(0)" ::: "memory"); TS(64 + blockIdx.x, 70); }
  if (tid == 0) __hip_atomic_fetch_add((gu32*)(myws + CTR_GEMM_OFF), 1u, __ATOMIC_RELAXED, __HIP_MEMORY_SCOPE_AGENT);
}

__global__ __launch_bounds__(512, 3) void fused_kernel(Args a) {
  extern __shared__ __attribute__((aligned(16))) char smem[];
  if (a.role == 0) comm_role(a);
  else gemm_role(a, smem);
}
}

extern "C" void kernel_launch(int rank, int nranks, ncclComm_t comm,
                              hipStream_t stream,
                              void* const* d_in, const int* in_sizes, int n_in,
                              void* d_out, int out_size,
                              const int* in_dtypes, int out_dtype,
                              void* const* d_out_all,
                              void* const* d_ws_all, size_t ws_bytes) {
  static hipStream_t side[64] = {};
  static hipEvent_t ev0[64] = {}, ev1[64] = {};
  int dev = 0;
  HIP_CHECK(hipGetDevice(&dev));
  if (dev < 0 || dev >= 64 || nranks != NR || ws_bytes < WS_NEEDED) { fprintf(stderr, "bad config\n"); return; }
  if (!side[dev]) {
    HIP_CHECK(hipStreamCreateWithFlags(&side[dev], hipStreamNonBlocking));
    HIP_CHECK(hipEventCreateWithFlags(&ev0[dev], hipEventDisableTiming));
    HIP_CHECK(hipEventCreateWithFlags(&ev1[dev], hipEventDisableTiming));
    HIP_CHECK(hipFuncSetAttribute((const void*)fused_kernel, hipFuncAttributeMaxDynamicSharedMemorySize, LDS_BYTES));
  }
  Args a;
  a.x = (const uint8_t*)d_in[0];
  a.w = (const uint8_t*)d_in[1];
  a.sx = (const float*)d_in[2];
  a.sw = (const float*)d_in[3];
  a.out = (float*)d_out;
  for (int p = 0; p < NR; ++p) a.ws[p] = (char*)d_ws_all[p];
  a.rank = rank;

  HIP_CHECK(hipEventRecord(ev0[dev], stream));
  HIP_CHECK(hipStreamWaitEvent(side[dev], ev0[dev], 0));
  a.role = 0;
  fused_kernel<<<2 * NCW, 64, 0, stream>>>(a);
  a.role = 1;
  fused_kernel<<<NGEMM, 512, LDS_BYTES, side[dev]>>>(a);
  HIP_CHECK(hipEventRecord(ev1[dev], side[dev]));
  HIP_CHECK(hipStreamWaitEvent(stream, ev1[dev], 0));
}
```

```cpp
#include <hip/hip_runtime.h>
#include <rccl/rccl.h>
#include <cstdio>
#include <cstdint>

#define HIP_CHECK(expr) do { hipError_t _e = (expr); if (_e != hipSuccess) { \
    fprintf(stderr, "HIP error %s at %s:%d\n", hipGetErrorString(_e), __FILE__, __LINE__); \
    return; } } while (0)

typedef int v8i __attribute__((ext_vector_type(8)));
typedef int v4i __attribute__((ext_vector_type(4)));
typedef unsigned v4u __attribute__((ext_vector_type(4)));
typedef float v16f __attribute__((ext_vector_type(16)));
typedef float v4f __attribute__((ext_vector_type(4)));
typedef __attribute__((address_space(1))) unsigned gu32;
typedef volatile __attribute__((address_space(3))) unsigned lu32;

namespace {
constexpr int MDIM = 4096, NDIM = 4096, KSH = 512, NR = 8;
constexpr int NSL = 16;
constexpr unsigned PIECE = 8192;
constexpr unsigned OPER_BYTES = 16 * PIECE;
constexpr unsigned SLICE_BYTES = 2 * OPER_BYTES;
constexpr unsigned SRC_BYTES = NSL * SLICE_BYTES;
constexpr size_t PAR_BYTES = (size_t)NR * SRC_BYTES;
constexpr size_t FLAG_OFF = 2 * PAR_BYTES;
constexpr size_t TICK_OFF = FLAG_OFF + NR * 128;
constexpr size_t RDY_OFF = TICK_OFF + 17 * 128;
constexpr size_t QUE_OFF = RDY_OFF + 8 * 128;
constexpr size_t CTR_OFF = QUE_OFF + 128;
constexpr size_t TMO_OFF = CTR_OFF + 128;
constexpr size_t WS_NEEDED = TMO_OFF + 128;
#define TS(slot, idx) do {} while (0)
constexpr int NGEMM = 256;
constexpr int NSTAGE = 64;
constexpr int RING = 4;
#define INFL 2
#define POLL_SLEEP 10
#define SPIN_SLEEP 2
constexpr unsigned STAGE_BYTES = 32768;
constexpr unsigned LDS_RING = RING * STAGE_BYTES;
constexpr unsigned LDS_BYTES = LDS_RING + 128;
constexpr unsigned long long DEADLINE_TICKS = 150000000ull;

struct Args {
  const uint8_t* x;
  const uint8_t* w;
  const float* sx;
  const float* sw;
  float* out;
  char* ws[NR];
  int rank;
  int mode;
};

__device__ __forceinline__ void entry_of(int n, int rank, int& src, int& q) {
  if (n < NSL) { src = rank; q = n; }
  else { int m = n - NSL; q = m / 7; src = (rank + 1 + (m - q * 7)) & 7; }
}

__device__ __forceinline__ unsigned ld_sys(const char* p) {
  return __hip_atomic_load((gu32*)p, __ATOMIC_RELAXED, __HIP_MEMORY_SCOPE_SYSTEM);
}
__device__ __forceinline__ unsigned ld_agent(const char* p) {
  return __hip_atomic_load((gu32*)p, __ATOMIC_RELAXED, __HIP_MEMORY_SCOPE_AGENT);
}

constexpr unsigned NUNIT = NSL * 256;
__device__ __forceinline__ v4u unit_fetch(const Args& a, unsigned v, int lane) {
  const unsigned q = v >> 8, u = (v & 127) >> 1, h = v & 1;
  v4u d;
  if ((v & 255) < 128) d = *(const v4u*)(a.x + (size_t)(u * 64 + lane) * KSH + h * 16 + q * 32);
  else {
    const uint8_t* src = a.w + (size_t)(q * 32 + h * 16 + (lane >> 4) * 4) * NDIM + u * 64 + (lane & 15) * 4;
#pragma unroll
    for (int i = 0; i < 4; ++i) d[i] = *(const unsigned*)(src + (size_t)i * NDIM);
  }
  return d;
}
__device__ __forceinline__ unsigned unit_pack(unsigned v, int lane, v4u& e) {
  const unsigned q = v >> 8, u = (v & 127) >> 1, h = v & 1;
  unsigned doff = q * SLICE_BYTES + (u >> 2) * PIECE + h * 4096 + (u & 3) * 1024;
  if ((v & 255) < 128) doff += OPER_BYTES + lane * 16;
  else {
    doff += ((lane & 15) * 4 + (lane >> 4)) * 16;
    unsigned t0 = __builtin_amdgcn_perm(e[1], e[0], 0x05010400u);
    unsigned t1 = __builtin_amdgcn_perm(e[1], e[0], 0x07030602u);
    unsigned t2 = __builtin_amdgcn_perm(e[3], e[2], 0x05010400u);
    unsigned t3 = __builtin_amdgcn_perm(e[3], e[2], 0x07030602u);
    unsigned e0 = __builtin_amdgcn_perm(t2, t0, 0x05040100u);
    unsigned e1 = __builtin_amdgcn_perm(t2, t0, 0x07060302u);
    unsigned e2 = __builtin_amdgcn_perm(t3, t1, 0x05040100u);
    unsigned e3 = __builtin_amdgcn_perm(t3, t1, 0x07060302u);
    auto s02 = __builtin_amdgcn_permlane32_swap(e0, e2, false, false);
    auto s13 = __builtin_amdgcn_permlane32_swap(e1, e3, false, false);
    auto s01 = __builtin_amdgcn_permlane16_swap(s02[0], s13[0], false, false);
    auto s23 = __builtin_amdgcn_permlane16_swap(s02[1], s13[1], false, false);
    e[0] = s01[0]; e[1] = s01[1]; e[2] = s23[0]; e[3] = s23[1];
  }
  return doff;
}

__device__ __forceinline__ void push_role(const Args& a, v4u e, unsigned call) {
  const int lane = threadIdx.x & 63;
  const int wg = blockIdx.x;
  const unsigned par = call & 1;
  char* myws = a.ws[a.rank];
  __amdgpu_buffer_rsrc_t rs[NR - 1];
#pragma unroll
  for (int k = 0; k < NR - 1; ++k) {
    int p = (a.rank + 1 + (k + wg) % 7) & 7;
    char* base = a.ws[p] + par * PAR_BYTES + (size_t)a.rank * SRC_BYTES;
    rs[k] = __builtin_amdgcn_make_buffer_rsrc(base, 0, SRC_BYTES, 0x00020000);
  }
  char* flagp = a.ws[(a.rank + 1 + lane) & 7] + FLAG_OFF + a.rank * 128;
  const unsigned qbase = (call - 1) * NUNIT - NGEMM;
  unsigned v = wg;
  while (true) {
    const unsigned doff = unit_pack(v, lane, e);
    unsigned tn = 0;
    if (lane == 0) tn = __hip_atomic_fetch_add((gu32*)(myws + QUE_OFF), 1u, __ATOMIC_RELAXED, __HIP_MEMORY_SCOPE_AGENT);
#pragma unroll
    for (int k = 0; k < NR - 1; ++k) {
      __builtin_amdgcn_raw_buffer_store_b128(e, rs[k], doff, 0, 17);
      if (k == INFL) asm volatile("s_waitcnt vmcnt(%0)" :: "i"(INFL) : "memory");
    }
    const unsigned vn = __builtin_amdgcn_readfirstlane(tn) - qbase;
    if (vn < NUNIT) e = unit_fetch(a, vn, lane);
    asm volatile("s_waitcnt vmcnt(0)" ::: "memory");
    const unsigned q = v >> 8;
    unsigned t = 0;
    if (lane == 0) t = __hip_atomic_fetch_add((gu32*)(myws + TICK_OFF + q * 128), 1u, __ATOMIC_RELAXED, __HIP_MEMORY_SCOPE_AGENT);
    t = __builtin_amdgcn_readfirstlane(t);
    if ((t & 255) == 255) {
      if (lane < NR - 1) __hip_atomic_store((gu32*)(flagp + q * 4), call, __ATOMIC_RELAXED, __HIP_MEMORY_SCOPE_SYSTEM);
      TS(0, 1 + q);
    }
    if (vn >= NUNIT) break;
    v = vn;
  }
}

__device__ __forceinline__ void self_role(const Args& a, v4u (&e)[NSL], unsigned call) {
  const int lane = threadIdx.x & 63;
  char* myws = a.ws[a.rank];
  __amdgpu_buffer_rsrc_t rs = __builtin_amdgcn_make_buffer_rsrc(
      myws + (call & 1) * PAR_BYTES + (size_t)a.rank * SRC_BYTES, 0, SRC_BYTES, 0x00020000);
#pragma unroll
  for (int q = 0; q < NSL; ++q) {
    const unsigned doff = unit_pack(q * 256 + blockIdx.x, lane, e[q]);
    __builtin_amdgcn_raw_buffer_store_b128(e[q], rs, doff, 0, 17);
  }
  asm volatile("s_waitcnt vmcnt(0)" ::: "memory");
  unsigned t = 0;
  if (lane == 0) t = __hip_atomic_fetch_add((gu32*)(myws + TICK_OFF + 16 * 128), 1u, __ATOMIC_RELAXED, __HIP_MEMORY_SCOPE_AGENT);
  t = __builtin_amdgcn_readfirstlane(t);
  if ((t & 255) == 255 && lane < NSL)
    __hip_atomic_store((gu32*)(myws + FLAG_OFF + a.rank * 128 + lane * 4), call, __ATOMIC_RELAXED, __HIP_MEMORY_SCOPE_SYSTEM);
}

__device__ __forceinline__ void gemm_role(const Args& a, char* smem, unsigned call, int wv) {
  const int lane = threadIdx.x & 63;
  const int rank = a.rank;
  char* myws = a.ws[rank];
  const unsigned par = call & 1;
  lu32* sync = (lu32*)(smem + LDS_RING);

  const int xcd = blockIdx.x & 7, ii = blockIdx.x >> 3;
  const int bn = (xcd >> 1) * 4 + (ii >> 3);
  const int bm = (xcd & 1) * 8 + (ii & 7);
  const int wn = wv >> 2, wm = wv & 3;

  const int ld_op = wv >> 2;
  const int ld_sl = (wv >> 1) & 1;
  const int ld_h = wv & 1;
  const unsigned voff = (ld_op ? OPER_BYTES + bm * PIECE : bn * PIECE) + ld_h * 4096 + lane * 16;
  constexpr unsigned gstep = 1024u;
  const char* stg = myws + par * PAR_BYTES;
  const unsigned lds0 = (unsigned)(size_t)smem + wv * 4096;

  auto issue = [&](int s) {
    int src, q;
    entry_of(2 * s + ld_sl, rank, src, q);
    const char* g = stg + (size_t)src * SRC_BYTES + (unsigned)q * SLICE_BYTES;
    const unsigned l = lds0 + (s & (RING - 1)) * STAGE_BYTES;
#pragma unroll
    for (int i = 0; i < 4; ++i)
      asm volatile("s_mov_b32 m0, %2\n\ts_nop 0\n\tglobal_load_lds_dwordx4 %0, %1"
                   :: "v"(voff), "s"(g + i * gstep), "s"(l + i * 1024) : "memory");
  };
  const int sl = lane < 16 ? lane : 16;
  auto wait_sync = [&](int dn, int ln, int rn) {
    const int need = lane < 8 ? dn : (lane < 16 ? ln : rn);
    while (true) {
      int v = (int)sync[sl];
      if (__all(v >= need)) break;
      __builtin_amdgcn_s_sleep(SPIN_SLEEP);
    }
    asm volatile("" ::: "memory");
  };

  const unsigned fa0 = (lane >> 5) * 8192 + (wn * 128 + (lane & 31)) * 16;
  const unsigned fb0 = 16384 + (lane >> 5) * 8192 + (wm * 64 + (lane & 31)) * 16;

  v16f acc[4][2];
#pragma unroll
  for (int i = 0; i < 4; ++i)
#pragma unroll
    for (int j = 0; j < 2; ++j)
#pragma unroll
      for (int e = 0; e < 16; ++e) acc[i][j][e] = 0.f;

  auto rd = [&](const char* base, unsigned off) -> v8i {
    v4i lo = *(const v4i*)(base + off);
    v4i hi = *(const v4i*)(base + off + 4096);
    return __builtin_shufflevector(lo, hi, 0, 1, 2, 3, 4, 5, 6, 7);
  };
#define MMA(c, x, y) c = __builtin_amdgcn_mfma_scale_f32_32x32x64_f8f6f4(x, y, c, 1, 1, 0, 0, 0, 0)
#define SB() __builtin_amdgcn_sched_barrier(0)

  wait_sync(0, 0, 3);
  issue(0); issue(1); issue(2);
  asm volatile("s_waitcnt vmcnt(8)" ::: "memory");
  if (lane == 0) sync[8 + wv] = 1;
  wait_sync(0, 1, 0);
  v8i a0 = rd(smem, fa0), b0 = rd(smem, fb0), b1 = rd(smem, fb0 + 512), a1 = rd(smem, fa0 + 512);

  for (int s = 0; s < NSTAGE; ++s) {
    const char* sb = smem + (s & (RING - 1)) * STAGE_BYTES;
    const char* sn = smem + ((s + 1) & (RING - 1)) * STAGE_BYTES;
    if (s + 2 < NSTAGE) asm volatile("s_waitcnt vmcnt(4)" ::: "memory");
    else asm volatile("s_waitcnt vmcnt(0)" ::: "memory");
    if (lane == 0) sync[8 + wv] = s + 2;
    SB();
    MMA(acc[0][0], a0, b0); MMA(acc[0][1], a0, b1);
    SB();
    a0 = rd(sb, fa0 + 1024);
    SB();
    MMA(acc[1][0], a1, b0); MMA(acc[1][1], a1, b1);
    SB();
    a1 = rd(sb, fa0 + 1536);
    SB();
    if (s + 3 < NSTAGE) { wait_sync(s, s + 2, s + 4); issue(s + 3); }
    else wait_sync(0, s + 2 < NSTAGE ? s + 2 : NSTAGE, 0);
    SB();
    MMA(acc[2][0], a0, b0); MMA(acc[2][1], a0, b1);
    SB();
    asm volatile("s_waitcnt lgkmcnt(0)" ::: "memory");
    if (lane == 0) sync[wv] = s + 1;
    if (s + 1 < NSTAGE) a0 = rd(sn, fa0);
    SB();
    MMA(acc[3][0], a1, b0);
    SB();
    if (s + 1 < NSTAGE) b0 = rd(sn, fb0);
    SB();
    MMA(acc[3][1], a1, b1);
    SB();
    if (s + 1 < NSTAGE) { b1 = rd(sn, fb0 + 512); a1 = rd(sn, fa0 + 512); }
    SB();
  }

  const bool bad = ld_agent(myws + TMO_OFF) == call;
  const float sc = a.sx[0] * a.sw[0];
  int tz = threadIdx.x;
  asm volatile("" : "+v"(tz));
  const int r = tz & 31, kg = (tz >> 5) & 1;
  float* op = a.out + (size_t)(bm * 256 + wm * 64 + r) * NDIM + bn * 256 + wn * 128 + kg * 4;
#pragma unroll
  for (int j = 0; j < 2; ++j)
#pragma unroll
    for (int i = 0; i < 4; ++i)
#pragma unroll
      for (int g = 0; g < 4; ++g) {
        v4f o;
#pragma unroll
        for (int e = 0; e < 4; ++e) {
          float y = acc[i][j][g * 4 + e] * sc;
          float c = fminf(fmaxf(y, -60.f), 60.f);
          float v = y * __builtin_amdgcn_rcpf(1.0f + __expf(-c));
          o[e] = bad ? __builtin_nanf("") : v;
        }
        *(v4f*)(op + (size_t)j * 32 * NDIM + i * 32 + g * 8) = o;
      }
}

__device__ __forceinline__ void poll_role(const Args& a, char* smem, unsigned call) {
  const int lane = threadIdx.x & 63;
  char* myws = a.ws[a.rank];
  lu32* sync = (lu32*)(smem + LDS_RING);
  const bool master = blockIdx.x == 0;
  int nready = 0;
  unsigned long long t0 = __builtin_amdgcn_s_memrealtime();
  for (unsigned it = 1;; ++it) {
    bool adv = false;
    if (master) {
      int n = nready + lane;
      bool ok = true;
      if (n < 2 * NSTAGE) {
        int src, q;
        entry_of(n, a.rank, src, q);
        ok = ld_sys(myws + FLAG_OFF + src * 128 + q * 4) >= call;
      }
      unsigned long long bal = __ballot(!ok);
      int lead = bal ? __builtin_ctzll(bal) : 64;
      if (lead) {
        nready = __builtin_amdgcn_readfirstlane(nready + lead);
        if (nready > 2 * NSTAGE) nready = 2 * NSTAGE;
        if (lane < 8) __hip_atomic_store((gu32*)(myws + RDY_OFF + lane * 128), (call << 8) | (unsigned)(nready >> 1),
                                         __ATOMIC_RELAXED, __HIP_MEMORY_SCOPE_AGENT);
        if (lane == 0) sync[16] = nready >> 1;
        TS(512, nready >> 1);
        if (nready >= 2 * NSTAGE) break;
        adv = true;
      }
    } else {
      unsigned v = __builtin_amdgcn_readfirstlane(ld_agent(myws + RDY_OFF + (blockIdx.x & 7) * 128));
      int n = (v >> 8) == call ? (int)(v & 255) : 0;
      if (n > nready) {
        nready = n;
        if (lane == 0) sync[16] = n;
        if (n >= NSTAGE) break;
      }
    }
    if (adv) continue;
    if ((it & 63) == 0 &&
        (ld_agent(myws + TMO_OFF) == call || __builtin_amdgcn_s_memrealtime() - t0 > DEADLINE_TICKS)) {
      if (lane == 0) {
        __hip_atomic_store((gu32*)(myws + TMO_OFF), call, __ATOMIC_RELAXED, __HIP_MEMORY_SCOPE_AGENT);
        asm volatile("s_waitcnt vmcnt(0)" ::: "memory");
        sync[16] = NSTAGE;
      }
      break;
    }
    __builtin_amdgcn_s_sleep(POLL_SLEEP);
  }
}

__global__ __launch_bounds__(640, 3) void fused_kernel(Args a) {
  extern __shared__ __attribute__((aligned(16))) char smem[];
  const int wv = __builtin_amdgcn_readfirstlane(threadIdx.x >> 6);
  const int lane = threadIdx.x & 63;
  char* myws = a.ws[a.rank];
  const unsigned cnt = ld_agent(myws + CTR_OFF);
  if (wv == 9) {
    __builtin_amdgcn_s_setprio(3);
    TS(blockIdx.x, 0);
    v4u e = unit_fetch(a, blockIdx.x, lane);
    __builtin_amdgcn_s_barrier();
    push_role(a, e, __builtin_amdgcn_readfirstlane(cnt) / NGEMM + 1);
    return;
  }
  if (wv == 8) {
    TS(256 + blockIdx.x, 0);
    if (lane < 32) ((lu32*)(smem + LDS_RING))[lane] = 0;
    asm volatile("s_waitcnt lgkmcnt(0)" ::: "memory");
    __builtin_amdgcn_s_barrier();
    const unsigned call = __builtin_amdgcn_readfirstlane(cnt) / NGEMM + 1;
    v4u e[NSL];
#pragma unroll
    for (int q = 0; q < NSL; ++q) e[q] = unit_fetch(a, q * 256 + blockIdx.x, lane);
    self_role(a, e, call);
    __builtin_amdgcn_fence(__ATOMIC_ACQUIRE, "");
    asm volatile("s_waitcnt vmcnt(0)" ::: "memory");
    poll_role(a, smem, call);
    return;
  }
  __builtin_amdgcn_s_barrier();
  gemm_role(a, smem, __builtin_amdgcn_readfirstlane(cnt) / NGEMM + 1, wv);
  if (threadIdx.x == 0) __hip_atomic_fetch_add((gu32*)(myws + CTR_OFF), 1u, __ATOMIC_RELAXED, __HIP_MEMORY_SCOPE_AGENT);
}
}

extern "C" void kernel_launch(int rank, int nranks, ncclComm_t comm,
                              hipStream_t stream,
                              void* const* d_in, const int* in_sizes, int n_in,
                              void* d_out, int out_size,
                              const int* in_dtypes, int out_dtype,
                              void* const* d_out_all,
                              void* const* d_ws_all, size_t ws_bytes) {
  static bool init[64] = {};
  int dev = 0;
  HIP_CHECK(hipGetDevice(&dev));
  if (dev < 0 || dev >= 64 || nranks != NR || ws_bytes < WS_NEEDED) { fprintf(stderr, "bad config\n"); return; }
  if (!init[dev]) {
    HIP_CHECK(hipFuncSetAttribute((const void*)fused_kernel, hipFuncAttributeMaxDynamicSharedMemorySize, LDS_BYTES));
    init[dev] = true;
  }
  Args a;
  a.x = (const uint8_t*)d_in[0];
  a.w = (const uint8_t*)d_in[1];
  a.sx = (const float*)d_in[2];
  a.sw = (const float*)d_in[3];
  a.out = (float*)d_out;
  for (int p = 0; p < NR; ++p) a.ws[p] = (char*)d_ws_all[p];
  a.rank = rank;

  a.mode = 0;
  fused_kernel<<<NGEMM, 640, LDS_BYTES, stream>>>(a);
}
```

```cpp
#include <hip/hip_runtime.h>
#include <rccl/rccl.h>
#include <cstdio>
#include <cstdint>

#define HIP_CHECK(expr) do { hipError_t _e = (expr); if (_e != hipSuccess) { \
    fprintf(stderr, "HIP error %s at %s:%d\n", hipGetErrorString(_e), __FILE__, __LINE__); \
    return; } } while (0)

typedef int v8i __attribute__((ext_vector_type(8)));
typedef int v4i __attribute__((ext_vector_type(4)));
typedef unsigned v4u __attribute__((ext_vector_type(4)));
typedef float v16f __attribute__((ext_vector_type(16)));
typedef float v4f __attribute__((ext_vector_type(4)));
typedef __attribute__((address_space(1))) unsigned gu32;
typedef volatile __attribute__((address_space(3))) unsigned lu32;

namespace {
constexpr int MDIM = 4096, NDIM = 4096, KSH = 512, NR = 8;
constexpr int NSL = 16;
constexpr unsigned PIECE = 8192;
constexpr unsigned OPER_BYTES = 16 * PIECE;
constexpr unsigned SLICE_BYTES = 2 * OPER_BYTES;
constexpr unsigned SRC_BYTES = NSL * SLICE_BYTES;
constexpr size_t PAR_BYTES = (size_t)NR * SRC_BYTES;
constexpr size_t FLAG_OFF = 2 * PAR_BYTES;
constexpr size_t TICK_OFF = FLAG_OFF + NR * 128;
constexpr size_t RDY_OFF = TICK_OFF + 17 * 128;
constexpr size_t QUE_OFF = RDY_OFF + 8 * 128;
constexpr size_t CTR_OFF = QUE_OFF + 128;
constexpr size_t TMO_OFF = CTR_OFF + 128;
constexpr size_t WS_NEEDED = TMO_OFF + 128;
#define TS(slot, idx) do {} while (0)
constexpr int NGEMM = 256;
constexpr int NSTAGE = 64;
constexpr int RING = 4;
#define OUT_MODE 0
#define PF_AHEAD 8
#define DIRECT_POLL 16
#define INFL 2
#define POLL_SLEEP 10
#define SPIN_SLEEP 2
constexpr unsigned STAGE_BYTES = 32768;
constexpr unsigned LDS_RING = RING * STAGE_BYTES;
constexpr unsigned LDS_BYTES = LDS_RING + 128;
constexpr unsigned long long DEADLINE_TICKS = 150000000ull;

struct Args {
  const uint8_t* x;
  const uint8_t* w;
  const float* sx;
  const float* sw;
  float* out;
  char* ws[NR];
  int rank;
  int mode;
};

__device__ __forceinline__ void entry_of(int n, int rank, int& src, int& q) {
  if (n < NSL) { src = rank; q = n; }
  else { int m = n - NSL; q = m / 7; src = (rank + 1 + (m - q * 7)) & 7; }
}

__device__ __forceinline__ unsigned ld_sys(const char* p) {
  return __hip_atomic_load((gu32*)p, __ATOMIC_RELAXED, __HIP_MEMORY_SCOPE_SYSTEM);
}
__device__ __forceinline__ unsigned ld_agent(const char* p) {
  return __hip_atomic_load((gu32*)p, __ATOMIC_RELAXED, __HIP_MEMORY_SCOPE_AGENT);
}

constexpr unsigned NUNIT = NSL * 256;
__device__ __forceinline__ v4u unit_fetch(const Args& a, unsigned v, int lane) {
  const unsigned q = v >> 8, u = (v & 127) >> 1, h = v & 1;
  v4u d;
  if ((v & 255) < 128) d = *(const v4u*)(a.x + (size_t)(u * 64 + lane) * KSH + h * 16 + q * 32);
  else {
    const uint8_t* src = a.w + (size_t)(q * 32 + h * 16 + (lane >> 4) * 4) * NDIM + u * 64 + (lane & 15) * 4;
#pragma unroll
    for (int i = 0; i < 4; ++i) d[i] = *(const unsigned*)(src + (size_t)i * NDIM);
  }
  return d;
}
__device__ __forceinline__ unsigned unit_pack(unsigned v, int lane, v4u& e) {
  const unsigned q = v >> 8, u = (v & 127) >> 1, h = v & 1;
  unsigned doff = q * SLICE_BYTES + (u >> 2) * PIECE + h * 4096 + (u & 3) * 1024;
  if ((v & 255) < 128) doff += OPER_BYTES + lane * 16;
  else {
    doff += ((lane & 15) * 4 + (lane >> 4)) * 16;
    unsigned t0 = __builtin_amdgcn_perm(e[1], e[0], 0x05010400u);
    unsigned t1 = __builtin_amdgcn_perm(e[1], e[0], 0x07030602u);
    unsigned t2 = __builtin_amdgcn_perm(e[3], e[2], 0x05010400u);
    unsigned t3 = __builtin_amdgcn_perm(e[3], e[2], 0x07030602u);
    unsigned e0 = __builtin_amdgcn_perm(t2, t0, 0x05040100u);
    unsigned e1 = __builtin_amdgcn_perm(t2, t0, 0x07060302u);
    unsigned e2 = __builtin_amdgcn_perm(t3, t1, 0x05040100u);
    unsigned e3 = __builtin_amdgcn_perm(t3, t1, 0x07060302u);
    auto s02 = __builtin_amdgcn_permlane32_swap(e0, e2, false, false);
    auto s13 = __builtin_amdgcn_permlane32_swap(e1, e3, false, false);
    auto s01 = __builtin_amdgcn_permlane16_swap(s02[0], s13[0], false, false);
    auto s23 = __builtin_amdgcn_permlane16_swap(s02[1], s13[1], false, false);
    e[0] = s01[0]; e[1] = s01[1]; e[2] = s23[0]; e[3] = s23[1];
  }
  return doff;
}

__device__ __forceinline__ void push_role(const Args& a, v4u e, unsigned call) {
  const int lane = threadIdx.x & 63;
  const int wg = blockIdx.x;
  const unsigned par = call & 1;
  char* myws = a.ws[a.rank];
  __amdgpu_buffer_rsrc_t rs[NR - 1];
#pragma unroll
  for (int k = 0; k < NR - 1; ++k) {
    int p = (a.rank + 1 + (k + wg) % 7) & 7;
    char* base = a.ws[p] + par * PAR_BYTES + (size_t)a.rank * SRC_BYTES;
    rs[k] = __builtin_amdgcn_make_buffer_rsrc(base, 0, SRC_BYTES, 0x00020000);
  }
  char* flagp = a.ws[(a.rank + 1 + lane) & 7] + FLAG_OFF + a.rank * 128;
  const unsigned qbase = (call - 1) * NUNIT - NGEMM;
  unsigned v = wg;
  TS(wg, 40);
  while (true) {
    const unsigned doff = unit_pack(v, lane, e);
    if (v == wg) TS(wg, 41);
    unsigned tn = 0;
    if (lane == 0) tn = __hip_atomic_fetch_add((gu32*)(myws + QUE_OFF), 1u, __ATOMIC_RELAXED, __HIP_MEMORY_SCOPE_AGENT);
#pragma unroll
    for (int k = 0; k < NR - 1; ++k) {
      __builtin_amdgcn_raw_buffer_store_b128(e, rs[k], doff, 0, 17);
      if (k >= INFL && k < NR - 2) asm volatile("s_waitcnt vmcnt(%0)" :: "i"(INFL) : "memory");
    }
    const unsigned vn = __builtin_amdgcn_readfirstlane(tn) - qbase;
    if (v == wg) TS(wg, 42);
    if (vn < NUNIT) e = unit_fetch(a, vn, lane);
    asm volatile("s_waitcnt vmcnt(0)" ::: "memory");
    if (v == wg) TS(wg, 43);
    const unsigned q = v >> 8;
    unsigned t = 0;
    if (lane == 0) t = __hip_atomic_fetch_add((gu32*)(myws + TICK_OFF + q * 128), 1u, __ATOMIC_RELAXED, __HIP_MEMORY_SCOPE_AGENT);
    t = __builtin_amdgcn_readfirstlane(t);
    if (v == wg) TS(wg, 44);
    if ((t & 255) == 255) {
      if (lane < NR - 1) __hip_atomic_store((gu32*)(flagp + q * 4), call, __ATOMIC_RELAXED, __HIP_MEMORY_SCOPE_SYSTEM);
      TS(0, 1 + q);
    }
    if (vn >= NUNIT) break;
    v = vn;
  }
}

__device__ __forceinline__ void self_role(const Args& a, v4u (&e)[NSL], unsigned call) {
  const int lane = threadIdx.x & 63;
  char* myws = a.ws[a.rank];
  __amdgpu_buffer_rsrc_t rs = __builtin_amdgcn_make_buffer_rsrc(
      myws + (call & 1) * PAR_BYTES + (size_t)a.rank * SRC_BYTES, 0, SRC_BYTES, 0x00020000);
#pragma unroll
  for (int q = 0; q < NSL; ++q) {
    const unsigned doff = unit_pack(q * 256 + blockIdx.x, lane, e[q]);
    __builtin_amdgcn_raw_buffer_store_b128(e[q], rs, doff, 0, 17);
  }
  TS(blockIdx.x, 45);
  asm volatile("s_waitcnt vmcnt(0)" ::: "memory");
  TS(blockIdx.x, 46);
  unsigned t = 0;
  if (lane == 0) t = __hip_atomic_fetch_add((gu32*)(myws + TICK_OFF + 16 * 128), 1u, __ATOMIC_RELAXED, __HIP_MEMORY_SCOPE_AGENT);
  t = __builtin_amdgcn_readfirstlane(t);
  if ((t & 255) == 255 && lane < NSL)
    __hip_atomic_store((gu32*)(myws + FLAG_OFF + a.rank * 128 + lane * 4), call, __ATOMIC_RELAXED, __HIP_MEMORY_SCOPE_SYSTEM);
}

__device__ __forceinline__ void gemm_role(const Args& a, char* smem, unsigned call, int wv) {
  const int lane = threadIdx.x & 63;
  const int rank = a.rank;
  char* myws = a.ws[rank];
  lu32* sync = (lu32*)(smem + LDS_RING);

  const int xcd = blockIdx.x & 7, ii = blockIdx.x >> 3;
  const int bn = (xcd >> 1) * 4 + (ii >> 3);
  const int bm = (xcd & 1) * 8 + (ii & 7);
  const int wn = wv >> 2, wm = wv & 3;

  auto wait_landed = [&](int ln) {
    while (true) {
      int v = (int)sync[8 + (lane & 1)];
      if (__all(v >= ln)) break;
      __builtin_amdgcn_s_sleep(SPIN_SLEEP);
    }
    asm volatile("" ::: "memory");
  };

  const unsigned fa0 = (lane >> 5) * 8192 + (wn * 128 + (lane & 31)) * 16;
  const unsigned fb0 = 16384 + (lane >> 5) * 8192 + (wm * 64 + (lane & 31)) * 16;

  v16f acc[4][2];
#pragma unroll
  for (int i = 0; i < 4; ++i)
#pragma unroll
    for (int j = 0; j < 2; ++j)
#pragma unroll
      for (int e = 0; e < 16; ++e) acc[i][j][e] = 0.f;

  auto rd = [&](const char* base, unsigned off) -> v8i {
    v4i lo = *(const v4i*)(base + off);
    v4i hi = *(const v4i*)(base + off + 4096);
    return __builtin_shufflevector(lo, hi, 0, 1, 2, 3, 4, 5, 6, 7);
  };
#define MMA(c, x, y) c = __builtin_amdgcn_mfma_scale_f32_32x32x64_f8f6f4(x, y, c, 1, 1, 0, 0, 0, 0)
#define SB() __builtin_amdgcn_sched_barrier(0)

  wait_landed(1);
  v8i a0 = rd(smem, fa0), b0 = rd(smem, fb0), b1 = rd(smem, fb0 + 512), a1 = rd(smem, fa0 + 512);

  for (int s = 0; s < NSTAGE; ++s) {
    const char* sb = smem + (s & (RING - 1)) * STAGE_BYTES;
    const char* sn = smem + ((s + 1) & (RING - 1)) * STAGE_BYTES;
    SB();
    MMA(acc[0][0], a0, b0); MMA(acc[0][1], a0, b1);
    SB();
    a0 = rd(sb, fa0 + 1024);
    SB();
    MMA(acc[1][0], a1, b0); MMA(acc[1][1], a1, b1);
    SB();
    a1 = rd(sb, fa0 + 1536);
    SB();
    if (s + 1 < NSTAGE) wait_landed(s + 2);
    SB();
    MMA(acc[2][0], a0, b0); MMA(acc[2][1], a0, b1);
    SB();
    asm volatile("s_waitcnt lgkmcnt(0)" ::: "memory");
    if (lane == 0) sync[wv] = s + 1;
    if (s + 1 < NSTAGE) a0 = rd(sn, fa0);
    SB();
    MMA(acc[3][0], a1, b0);
    SB();
    if (s + 1 < NSTAGE) b0 = rd(sn, fb0);
    SB();
    MMA(acc[3][1], a1, b1);
    SB();
    if (s + 1 < NSTAGE) { b1 = rd(sn, fb0 + 512); a1 = rd(sn, fa0 + 512); }
    SB();
  }

  const bool bad = ld_agent(myws + TMO_OFF) == call;
  const float sc = a.sx[0] * a.sw[0];
  int tz = threadIdx.x;
  asm volatile("" : "+v"(tz));
  const int r = tz & 31, kg = (tz >> 5) & 1;
  float* op = a.out + (size_t)(bm * 256 + wm * 64 + r) * NDIM + bn * 256 + wn * 128 + kg * 4;
#pragma unroll
  for (int j = 0; j < 2; ++j)
#pragma unroll
    for (int i = 0; i < 4; ++i)
#pragma unroll
      for (int g = 0; g < 4; ++g) {
        v4f o;
#pragma unroll
        for (int e = 0; e < 4; ++e) {
          float y = acc[i][j][g * 4 + e] * sc;
          float c = fminf(fmaxf(y, -60.f), 60.f);
          float v = y * __builtin_amdgcn_rcpf(1.0f + __expf(-c));
          o[e] = bad ? __builtin_nanf("") : v;
        }
        *(v4f*)(op + (size_t)j * 32 * NDIM + i * 32 + g * 8) = o;
      }
}

__device__ __forceinline__ void poll_role(const Args& a, char* smem, unsigned call) {
  const int lane = threadIdx.x & 63;
  char* myws = a.ws[a.rank];
  lu32* sync = (lu32*)(smem + LDS_RING);
  const bool master = true;
  int nready = 0;
  unsigned long long t0 = __builtin_amdgcn_s_memrealtime();
  for (unsigned it = 1;; ++it) {
    bool adv = false;
    if (master) {
      int n = nready + lane;
      bool ok = true;
      constexpr int LOOK = DIRECT_POLL;
      if (lane < LOOK && n < 2 * NSTAGE) {
        int src, q;
        entry_of(n, a.rank, src, q);
        ok = ld_sys(myws + FLAG_OFF + src * 128 + q * 4) >= call;
      }
      unsigned long long bal = __ballot(!ok);
      int lead = bal ? __builtin_ctzll(bal) : 64;
      if (lead > LOOK) lead = LOOK;
      if (lead) {
        nready = __builtin_amdgcn_readfirstlane(nready + lead);
        if (nready > 2 * NSTAGE) nready = 2 * NSTAGE;
        if (lane == 0) sync[16] = nready >> 1;
        if (blockIdx.x == 0) TS(512, nready >> 1);
        if (nready >= 2 * NSTAGE) break;
        adv = true;
      }
    } else {
      unsigned v = __builtin_amdgcn_readfirstlane(ld_agent(myws + RDY_OFF + (blockIdx.x & 7) * 128));
      int n = (v >> 8) == call ? (int)(v & 255) : 0;
      if (n > nready) {
        nready = n;
        if (lane == 0) sync[16] = n;
        if (n >= NSTAGE) break;
      }
    }
    if (adv) continue;
    if ((it & 63) == 0 &&
        (ld_agent(myws + TMO_OFF) == call || __builtin_amdgcn_s_memrealtime() - t0 > DEADLINE_TICKS)) {
      if (lane == 0) {
        __hip_atomic_store((gu32*)(myws + TMO_OFF), call, __ATOMIC_RELAXED, __HIP_MEMORY_SCOPE_AGENT);
        asm volatile("s_waitcnt vmcnt(0)" ::: "memory");
        sync[16] = NSTAGE;
      }
      break;
    }
    __builtin_amdgcn_s_sleep(POLL_SLEEP);
  }
}

__device__ __forceinline__ void load_role(const Args& a, char* smem, unsigned call, int op) {
  const int lane = threadIdx.x & 63;
  lu32* sync = (lu32*)(smem + LDS_RING);
  const int xcd = blockIdx.x & 7, ii = blockIdx.x >> 3;
  const int bn = (xcd >> 1) * 4 + (ii >> 3), bm = (xcd & 1) * 8 + (ii & 7);
  const unsigned voff = (op ? OPER_BYTES + bm * PIECE : bn * PIECE) + lane * 16;
  const char* stg = a.ws[a.rank] + (call & 1) * PAR_BYTES;
  const unsigned lds0 = (unsigned)(size_t)smem + op * 16384;
  const int sidx = lane < 8 ? lane : 16;
  auto can_issue = [&](int p) -> bool {
    int v = (int)sync[sidx];
    return __all(v >= (lane < 8 ? p - (RING - 1) : p + 1));
  };
  int pub = 0;
  for (int p = 0; p < NSTAGE; ++p) {
    if (!can_issue(p)) {
      asm volatile("s_waitcnt vmcnt(0)" ::: "memory");
      if (lane == 0) sync[8 + op] = p;
      pub = p;
      while (!can_issue(p)) __builtin_amdgcn_s_sleep(SPIN_SLEEP);
    }
    asm volatile("" ::: "memory");
#pragma unroll
    for (int sl = 0; sl < 2; ++sl) {
      int src, q;
      entry_of(2 * p + sl, a.rank, src, q);
      const char* g = stg + (size_t)src * SRC_BYTES + (unsigned)q * SLICE_BYTES;
      const unsigned l = lds0 + (p & (RING - 1)) * STAGE_BYTES + sl * 8192;
#pragma unroll
      for (int i = 0; i < 8; ++i)
        asm volatile("s_mov_b32 m0, %2\n\ts_nop 0\n\tglobal_load_lds_dwordx4 %0, %1"
                     :: "v"(voff), "s"(g + i * 1024), "s"(l + i * 1024) : "memory");
    }
    asm volatile("s_waitcnt vmcnt(32)" ::: "memory");
    if (p - 1 > pub) { pub = p - 1; if (lane == 0) sync[8 + op] = p - 1; }
  }
  asm volatile("s_waitcnt vmcnt(16)" ::: "memory");
  if (lane == 0) sync[8 + op] = NSTAGE - 1;
  asm volatile("s_waitcnt vmcnt(0)" ::: "memory");
  if (lane == 0) sync[8 + op] = NSTAGE;
}

__global__ __launch_bounds__(768, 3) void fused_kernel(Args a) {
  extern __shared__ __attribute__((aligned(16))) char smem[];
  const int wv = __builtin_amdgcn_readfirstlane(threadIdx.x >> 6);
  const int lane = threadIdx.x & 63;
  char* myws = a.ws[a.rank];
  const unsigned cnt = ld_agent(myws + CTR_OFF);
  if (wv == 9) {
    __builtin_amdgcn_s_setprio(3);
    TS(blockIdx.x, 0);
    v4u e = unit_fetch(a, blockIdx.x, lane);
    __builtin_amdgcn_s_barrier();
    push_role(a, e, __builtin_amdgcn_readfirstlane(cnt) / NGEMM + 1);
    return;
  }
  if (wv == 8) {
    TS(256 + blockIdx.x, 0);
    if (lane < 32) ((lu32*)(smem + LDS_RING))[lane] = 0;
    asm volatile("s_waitcnt lgkmcnt(0)" ::: "memory");
    __builtin_amdgcn_s_barrier();
    const unsigned call = __builtin_amdgcn_readfirstlane(cnt) / NGEMM + 1;
    v4u e[NSL];
#pragma unroll
    for (int q = 0; q < NSL; ++q) e[q] = unit_fetch(a, q * 256 + blockIdx.x, lane);
    self_role(a, e, call);
    __builtin_amdgcn_fence(__ATOMIC_ACQUIRE, "");
    asm volatile("s_waitcnt vmcnt(0)" ::: "memory");
    poll_role(a, smem, call);
    return;
  }
  __builtin_amdgcn_s_barrier();
  if (wv >= 10) {
    load_role(a, smem, __builtin_amdgcn_readfirstlane(cnt) / NGEMM + 1, wv - 10);
    return;
  }
  gemm_role(a, smem, __builtin_amdgcn_readfirstlane(cnt) / NGEMM + 1, wv);
  if (threadIdx.x == 0) __hip_atomic_fetch_add((gu32*)(myws + CTR_OFF), 1u, __ATOMIC_RELAXED, __HIP_MEMORY_SCOPE_AGENT);
}
}

extern "C" void kernel_launch(int rank, int nranks, ncclComm_t comm,
                              hipStream_t stream,
                              void* const* d_in, const int* in_sizes, int n_in,
                              void* d_out, int out_size,
                              const int* in_dtypes, int out_dtype,
                              void* const* d_out_all,
                              void* const* d_ws_all, size_t ws_bytes) {
  static bool init[64] = {};
  int dev = 0;
  HIP_CHECK(hipGetDevice(&dev));
  if (dev < 0 || dev >= 64 || nranks != NR || ws_bytes < WS_NEEDED) { fprintf(stderr, "bad config\n"); return; }
  if (!init[dev]) {
    HIP_CHECK(hipFuncSetAttribute((const void*)fused_kernel, hipFuncAttributeMaxDynamicSharedMemorySize, LDS_BYTES));
    init[dev] = true;
  }
  Args a;
  a.x = (const uint8_t*)d_in[0];
  a.w = (const uint8_t*)d_in[1];
  a.sx = (const float*)d_in[2];
  a.sw = (const float*)d_in[3];
  a.out = (float*)d_out;
  for (int p = 0; p < NR; ++p) a.ws[p] = (char*)d_ws_all[p];
  a.rank = rank;

  a.mode = 0;
  fused_kernel<<<NGEMM, 768, LDS_BYTES, stream>>>(a);
}
```

```cpp
#include <hip/hip_runtime.h>
#include <rccl/rccl.h>
#include <cstdio>
#include <cstdint>

#define HIP_CHECK(expr) do { hipError_t _e = (expr); if (_e != hipSuccess) { \
    fprintf(stderr, "HIP error %s at %s:%d\n", hipGetErrorString(_e), __FILE__, __LINE__); \
    return; } } while (0)

typedef int v8i __attribute__((ext_vector_type(8)));
typedef int v4i __attribute__((ext_vector_type(4)));
typedef unsigned v4u __attribute__((ext_vector_type(4)));
typedef float v16f __attribute__((ext_vector_type(16)));
typedef float v4f __attribute__((ext_vector_type(4)));
typedef __attribute__((address_space(1))) unsigned gu32;
typedef volatile __attribute__((address_space(3))) unsigned lu32;

namespace {
constexpr int MDIM = 4096, NDIM = 4096, KSH = 512, NR = 8;
constexpr int NSL = 16;
constexpr unsigned PIECE = 8192;
constexpr unsigned OPER_BYTES = 16 * PIECE;
constexpr unsigned SLICE_BYTES = 2 * OPER_BYTES;
constexpr unsigned SRC_BYTES = NSL * SLICE_BYTES;
constexpr size_t PAR_BYTES = (size_t)NR * SRC_BYTES;
constexpr size_t FLAG_OFF = 2 * PAR_BYTES;
constexpr size_t TICK_OFF = FLAG_OFF + NR * 128;
constexpr size_t QUE_OFF = TICK_OFF + 17 * 128;
constexpr size_t CTR_OFF = QUE_OFF + 128;
constexpr size_t TMO_OFF = CTR_OFF + 128;
constexpr size_t WS_NEEDED = TMO_OFF + 128;
#define TS(slot, idx) do {} while (0)
constexpr int NGEMM = 256;
constexpr int NSTAGE = 64;
constexpr int RING = 4;
constexpr int INFL = 2;
constexpr int LOOK = 16;
constexpr int POLL_SLEEP = 10, SPIN_SLEEP = 2;
constexpr unsigned STAGE_BYTES = 32768;
constexpr unsigned LDS_RING = RING * STAGE_BYTES;
constexpr unsigned LDS_BYTES = LDS_RING + 128;
constexpr unsigned long long DEADLINE_TICKS = 150000000ull;

struct Args {
  const uint8_t* x;
  const uint8_t* w;
  const float* sx;
  const float* sw;
  float* out;
  char* ws[NR];
  int rank;
};

__device__ __forceinline__ void entry_of(int n, int rank, int& src, int& q) {
  if (n < NSL) { src = rank; q = n; }
  else { int m = n - NSL; q = m / 7; src = (rank + 1 + (m - q * 7)) & 7; }
}

__device__ __forceinline__ unsigned ld_sys(const char* p) {
  return __hip_atomic_load((gu32*)p, __ATOMIC_RELAXED, __HIP_MEMORY_SCOPE_SYSTEM);
}
__device__ __forceinline__ unsigned ld_agent(const char* p) {
  return __hip_atomic_load((gu32*)p, __ATOMIC_RELAXED, __HIP_MEMORY_SCOPE_AGENT);
}

constexpr unsigned NUNIT = NSL * 256;
__device__ __forceinline__ v4u unit_fetch(const Args& a, unsigned v, int lane) {
  const unsigned q = v >> 8, u = (v & 127) >> 1, h = v & 1;
  v4u d;
  if ((v & 255) < 128) d = *(const v4u*)(a.x + (size_t)(u * 64 + lane) * KSH + h * 16 + q * 32);
  else {
    const uint8_t* src = a.w + (size_t)(q * 32 + h * 16 + (lane >> 4) * 4) * NDIM + u * 64 + (lane & 15) * 4;
#pragma unroll
    for (int i = 0; i < 4; ++i) d[i] = *(const unsigned*)(src + (size_t)i * NDIM);
  }
  return d;
}
__device__ __forceinline__ unsigned unit_pack(unsigned v, int lane, v4u& e) {
  const unsigned q = v >> 8, u = (v & 127) >> 1, h = v & 1;
  unsigned doff = q * SLICE_BYTES + (u >> 2) * PIECE + h * 4096 + (u & 3) * 1024;
  if ((v & 255) < 128) doff += OPER_BYTES + lane * 16;
  else {
    doff += ((lane & 15) * 4 + (lane >> 4)) * 16;
    unsigned t0 = __builtin_amdgcn_perm(e[1], e[0], 0x05010400u);
    unsigned t1 = __builtin_amdgcn_perm(e[1], e[0], 0x07030602u);
    unsigned t2 = __builtin_amdgcn_perm(e[3], e[2], 0x05010400u);
    unsigned t3 = __builtin_amdgcn_perm(e[3], e[2], 0x07030602u);
    unsigned e0 = __builtin_amdgcn_perm(t2, t0, 0x05040100u);
    unsigned e1 = __builtin_amdgcn_perm(t2, t0, 0x07060302u);
    unsigned e2 = __builtin_amdgcn_perm(t3, t1, 0x05040100u);
    unsigned e3 = __builtin_amdgcn_perm(t3, t1, 0x07060302u);
    auto s02 = __builtin_amdgcn_permlane32_swap(e0, e2, false, false);
    auto s13 = __builtin_amdgcn_permlane32_swap(e1, e3, false, false);
    auto s01 = __builtin_amdgcn_permlane16_swap(s02[0], s13[0], false, false);
    auto s23 = __builtin_amdgcn_permlane16_swap(s02[1], s13[1], false, false);
    e[0] = s01[0]; e[1] = s01[1]; e[2] = s23[0]; e[3] = s23[1];
  }
  return doff;
}

__device__ __forceinline__ void push_role(const Args& a, v4u e, unsigned call) {
  const int lane = threadIdx.x & 63;
  const int wg = blockIdx.x;
  const unsigned par = call & 1;
  char* myws = a.ws[a.rank];
  __amdgpu_buffer_rsrc_t rs[NR - 1];
#pragma unroll
  for (int k = 0; k < NR - 1; ++k) {
    int p = (a.rank + 1 + (k + wg) % 7) & 7;
    char* base = a.ws[p] + par * PAR_BYTES + (size_t)a.rank * SRC_BYTES;
    rs[k] = __builtin_amdgcn_make_buffer_rsrc(base, 0, SRC_BYTES, 0x00020000);
  }
  char* flagp = a.ws[(a.rank + 1 + lane) & 7] + FLAG_OFF + a.rank * 128;
  const unsigned qbase = (call - 1) * NUNIT - NGEMM;
  unsigned v = wg;
  while (true) {
    const unsigned doff = unit_pack(v, lane, e);
    unsigned tn = 0;
    if (lane == 0) tn = __hip_atomic_fetch_add((gu32*)(myws + QUE_OFF), 1u, __ATOMIC_RELAXED, __HIP_MEMORY_SCOPE_AGENT);
#pragma unroll
    for (int k = 0; k < NR - 1; ++k) {
      __builtin_amdgcn_raw_buffer_store_b128(e, rs[k], doff, 0, 17);
      if (k >= INFL && k < NR - 2) asm volatile("s_waitcnt vmcnt(%0)" :: "i"(INFL) : "memory");
    }
    const unsigned vn = __builtin_amdgcn_readfirstlane(tn) - qbase;
    if (vn < NUNIT) e = unit_fetch(a, vn, lane);
    asm volatile("s_waitcnt vmcnt(0)" ::: "memory");
    const unsigned q = v >> 8;
    unsigned t = 0;
    if (lane == 0) t = __hip_atomic_fetch_add((gu32*)(myws + TICK_OFF + q * 128), 1u, __ATOMIC_RELAXED, __HIP_MEMORY_SCOPE_AGENT);
    t = __builtin_amdgcn_readfirstlane(t);
    if ((t & 255) == 255) {
      if (lane < NR - 1) __hip_atomic_store((gu32*)(flagp + q * 4), call, __ATOMIC_RELAXED, __HIP_MEMORY_SCOPE_SYSTEM);
      TS(0, 1 + q);
    }
    if (vn >= NUNIT) break;
    v = vn;
  }
}

__device__ __forceinline__ void self_role(const Args& a, v4u (&e)[NSL], unsigned call) {
  const int lane = threadIdx.x & 63;
  char* myws = a.ws[a.rank];
  __amdgpu_buffer_rsrc_t rs = __builtin_amdgcn_make_buffer_rsrc(
      myws + (call & 1) * PAR_BYTES + (size_t)a.rank * SRC_BYTES, 0, SRC_BYTES, 0x00020000);
#pragma unroll
  for (int q = 0; q < NSL; ++q) {
    const unsigned doff = unit_pack(q * 256 + blockIdx.x, lane, e[q]);
    __builtin_amdgcn_raw_buffer_store_b128(e[q], rs, doff, 0, 17);
  }
  asm volatile("s_waitcnt vmcnt(0)" ::: "memory");
  unsigned t = 0;
  if (lane == 0) t = __hip_atomic_fetch_add((gu32*)(myws + TICK_OFF + 16 * 128), 1u, __ATOMIC_RELAXED, __HIP_MEMORY_SCOPE_AGENT);
  t = __builtin_amdgcn_readfirstlane(t);
  if ((t & 255) == 255 && lane < NSL)
    __hip_atomic_store((gu32*)(myws + FLAG_OFF + a.rank * 128 + lane * 4), call, __ATOMIC_RELAXED, __HIP_MEMORY_SCOPE_SYSTEM);
}

__device__ __forceinline__ void gemm_role(const Args& a, char* smem, unsigned call, int wv) {
  const int lane = threadIdx.x & 63;
  const int rank = a.rank;
  char* myws = a.ws[rank];
  lu32* sync = (lu32*)(smem + LDS_RING);

  const int xcd = blockIdx.x & 7, ii = blockIdx.x >> 3;
  const int bn = (xcd >> 1) * 4 + (ii >> 3);
  const int bm = (xcd & 1) * 8 + (ii & 7);
  const int wn = wv >> 2, wm = wv & 3;

  auto wait_landed = [&](int ln) {
    while (true) {
      int v = (int)sync[8 + (lane & 1)];
      if (__all(v >= ln)) break;
      __builtin_amdgcn_s_sleep(SPIN_SLEEP);
    }
    asm volatile("" ::: "memory");
  };

  const unsigned fa0 = (lane >> 5) * 8192 + (wn * 128 + (lane & 31)) * 16;
  const unsigned fb0 = 16384 + (lane >> 5) * 8192 + (wm * 64 + (lane & 31)) * 16;

  v16f acc[4][2];
#pragma unroll
  for (int i = 0; i < 4; ++i)
#pragma unroll
    for (int j = 0; j < 2; ++j)
#pragma unroll
      for (int e = 0; e < 16; ++e) acc[i][j][e] = 0.f;

  auto rd = [&](const char* base, unsigned off) -> v8i {
    v4i lo = *(const v4i*)(base + off);
    v4i hi = *(const v4i*)(base + off + 4096);
    return __builtin_shufflevector(lo, hi, 0, 1, 2, 3, 4, 5, 6, 7);
  };
#define MMA(c, x, y) c = __builtin_amdgcn_mfma_scale_f32_32x32x64_f8f6f4(x, y, c, 1, 1, 0, 0, 0, 0)
#define SB() __builtin_amdgcn_sched_barrier(0)

  wait_landed(1);
  v8i a0 = rd(smem, fa0), b0 = rd(smem, fb0), b1 = rd(smem, fb0 + 512), a1 = rd(smem, fa0 + 512);

  for (int s = 0; s < NSTAGE; ++s) {
    const char* sb = smem + (s & (RING - 1)) * STAGE_BYTES;
    const char* sn = smem + ((s + 1) & (RING - 1)) * STAGE_BYTES;
    SB();
    MMA(acc[0][0], a0, b0); MMA(acc[0][1], a0, b1);
    SB();
    a0 = rd(sb, fa0 + 1024);
    SB();
    MMA(acc[1][0], a1, b0); MMA(acc[1][1], a1, b1);
    SB();
    a1 = rd(sb, fa0 + 1536);
    SB();
    if (s + 1 < NSTAGE) wait_landed(s + 2);
    SB();
    MMA(acc[2][0], a0, b0); MMA(acc[2][1], a0, b1);
    SB();
    asm volatile("s_waitcnt lgkmcnt(0)" ::: "memory");
    if (lane == 0) sync[wv] = s + 1;
    if (s + 1 < NSTAGE) a0 = rd(sn, fa0);
    SB();
    MMA(acc[3][0], a1, b0);
    SB();
    if (s + 1 < NSTAGE) b0 = rd(sn, fb0);
    SB();
    MMA(acc[3][1], a1, b1);
    SB();
    if (s + 1 < NSTAGE) { b1 = rd(sn, fb0 + 512); a1 = rd(sn, fa0 + 512); }
    SB();
  }

  const bool bad = ld_agent(myws + TMO_OFF) == call;
  const float sc = a.sx[0] * a.sw[0];
  const int tz = __lane_id();
  const int r = tz & 31, kg = tz >> 5;
  float* op = a.out + (size_t)(bm * 256 + wm * 64 + r) * NDIM + bn * 256 + wn * 128 + kg * 4;
#pragma unroll
  for (int j = 0; j < 2; ++j)
#pragma unroll
    for (int i = 0; i < 4; ++i)
#pragma unroll
      for (int g = 0; g < 4; ++g) {
        v4f o;
#pragma unroll
        for (int e = 0; e < 4; ++e) {
          float y = acc[i][j][g * 4 + e] * sc;
          float c = fminf(fmaxf(y, -60.f), 60.f);
          float v = y * __builtin_amdgcn_rcpf(1.0f + __expf(-c));
          o[e] = bad ? __builtin_nanf("") : v;
        }
        *(v4f*)(op + (size_t)j * 32 * NDIM + i * 32 + g * 8) = o;
      }
}

__device__ __forceinline__ void poll_role(const Args& a, char* smem, unsigned call) {
  const int lane = threadIdx.x & 63;
  char* myws = a.ws[a.rank];
  lu32* sync = (lu32*)(smem + LDS_RING);
  int nready = 0;
  unsigned long long t0 = __builtin_amdgcn_s_memrealtime();
  for (unsigned it = 1;; ++it) {
    int n = nready + lane;
    bool ok = true;
    if (lane < LOOK && n < 2 * NSTAGE) {
      int src, q;
      entry_of(n, a.rank, src, q);
      ok = ld_sys(myws + FLAG_OFF + src * 128 + q * 4) >= call;
    }
    unsigned long long bal = __ballot(!ok);
    int lead = bal ? __builtin_ctzll(bal) : 64;
    if (lead > LOOK) lead = LOOK;
    if (lead) {
      nready = __builtin_amdgcn_readfirstlane(nready + lead);
      if (nready > 2 * NSTAGE) nready = 2 * NSTAGE;
      if (lane == 0) sync[16] = nready >> 1;
      if (blockIdx.x == 0) TS(512, nready >> 1);
      if (nready >= 2 * NSTAGE) break;
      continue;
    }
    if ((it & 63) == 0 &&
        (ld_agent(myws + TMO_OFF) == call || __builtin_amdgcn_s_memrealtime() - t0 > DEADLINE_TICKS)) {
      if (lane == 0) {
        __hip_atomic_store((gu32*)(myws + TMO_OFF), call, __ATOMIC_RELAXED, __HIP_MEMORY_SCOPE_AGENT);
        asm volatile("s_waitcnt vmcnt(0)" ::: "memory");
        sync[16] = NSTAGE;
      }
      break;
    }
    __builtin_amdgcn_s_sleep(POLL_SLEEP);
  }
}

__device__ __forceinline__ void load_role(const Args& a, char* smem, unsigned call, int op) {
  const int lane = threadIdx.x & 63;
  lu32* sync = (lu32*)(smem + LDS_RING);
  const int xcd = blockIdx.x & 7, ii = blockIdx.x >> 3;
  const int bn = (xcd >> 1) * 4 + (ii >> 3), bm = (xcd & 1) * 8 + (ii & 7);
  const unsigned voff = (op ? OPER_BYTES + bm * PIECE : bn * PIECE) + lane * 16;
  const char* stg = a.ws[a.rank] + (call & 1) * PAR_BYTES;
  const unsigned lds0 = (unsigned)(size_t)smem + op * 16384;
  const int sidx = lane < 8 ? lane : 16;
  auto can_issue = [&](int p) -> bool {
    int v = (int)sync[sidx];
    return __all(v >= (lane < 8 ? p - (RING - 1) : p + 1));
  };
  int pub = 0;
  for (int p = 0; p < NSTAGE; ++p) {
    if (!can_issue(p)) {
      asm volatile("s_waitcnt vmcnt(0)" ::: "memory");
      if (lane == 0) sync[8 + op] = p;
      pub = p;
      while (!can_issue(p)) __builtin_amdgcn_s_sleep(SPIN_SLEEP);
    }
    asm volatile("" ::: "memory");
#pragma unroll
    for (int sl = 0; sl < 2; ++sl) {
      int src, q;
      entry_of(2 * p + sl, a.rank, src, q);
      const char* g = stg + (size_t)src * SRC_BYTES + (unsigned)q * SLICE_BYTES;
      const unsigned l = lds0 + (p & (RING - 1)) * STAGE_BYTES + sl * 8192;
#pragma unroll
      for (int i = 0; i < 8; ++i)
        asm volatile("s_mov_b32 m0, %2\n\ts_nop 0\n\tglobal_load_lds_dwordx4 %0, %1"
                     :: "v"(voff), "s"(g + i * 1024), "s"(l + i * 1024) : "memory");
    }
    asm volatile("s_waitcnt vmcnt(32)" ::: "memory");
    if (p - 1 > pub) { pub = p - 1; if (lane == 0) sync[8 + op] = p - 1; }
  }
  asm volatile("s_waitcnt vmcnt(16)" ::: "memory");
  if (lane == 0) sync[8 + op] = NSTAGE - 1;
  asm volatile("s_waitcnt vmcnt(0)" ::: "memory");
  if (lane == 0) sync[8 + op] = NSTAGE;
}

__global__ __launch_bounds__(768, 3) void fused_kernel(Args a) {
  extern __shared__ __attribute__((aligned(16))) char smem[];
  const int wv = __builtin_amdgcn_readfirstlane(threadIdx.x >> 6);
  const int lane = threadIdx.x & 63;
  char* myws = a.ws[a.rank];
  if (wv == 9) {
    const unsigned cnt = ld_agent(myws + CTR_OFF);
    v4u e = unit_fetch(a, blockIdx.x, lane);
    __builtin_amdgcn_s_barrier();
    __builtin_amdgcn_s_setprio(3);
    TS(blockIdx.x, 0);
    push_role(a, e, __builtin_amdgcn_readfirstlane(cnt) / NGEMM + 1);
    return;
  }
  if (wv == 8 && lane < 32) ((lu32*)(smem + LDS_RING))[lane] = 0;
  const unsigned call = __builtin_amdgcn_readfirstlane(ld_agent(myws + CTR_OFF)) / NGEMM + 1;
  asm volatile("s_waitcnt lgkmcnt(0)" ::: "memory");
  __builtin_amdgcn_s_barrier();
  if (wv == 8) {
    TS(256 + blockIdx.x, 0);
    v4u e[NSL];
#pragma unroll
    for (int q = 0; q < NSL; ++q) e[q] = unit_fetch(a, q * 256 + blockIdx.x, lane);
    self_role(a, e, call);
    __builtin_amdgcn_fence(__ATOMIC_ACQUIRE, "");
    asm volatile("s_waitcnt vmcnt(0)" ::: "memory");
    poll_role(a, smem, call);
    return;
  }
  if (wv >= 10) {
    load_role(a, smem, call, wv - 10);
    return;
  }
  gemm_role(a, smem, call, wv);
  if (wv == 0 && __lane_id() == 0)
    __hip_atomic_fetch_add((gu32*)(myws + CTR_OFF), 1u, __ATOMIC_RELAXED, __HIP_MEMORY_SCOPE_AGENT);
}
}

extern "C" void kernel_launch(int rank, int nranks, ncclComm_t comm,
                              hipStream_t stream,
                              void* const* d_in, const int* in_sizes, int n_in,
                              void* d_out, int out_size,
                              const int* in_dtypes, int out_dtype,
                              void* const* d_out_all,
                              void* const* d_ws_all, size_t ws_bytes) {
  static bool init[64] = {};
  int dev = 0;
  HIP_CHECK(hipGetDevice(&dev));
  if (dev < 0 || dev >= 64 || nranks != NR || ws_bytes < WS_NEEDED) { fprintf(stderr, "bad config\n"); return; }
  if (!init[dev]) {
    HIP_CHECK(hipFuncSetAttribute((const void*)fused_kernel, hipFuncAttributeMaxDynamicSharedMemorySize, LDS_BYTES));
    init[dev] = true;
  }
  Args a;
  a.x = (const uint8_t*)d_in[0];
  a.w = (const uint8_t*)d_in[1];
  a.sx = (const float*)d_in[2];
  a.sw = (const float*)d_in[3];
  a.out = (float*)d_out;
  for (int p = 0; p < NR; ++p) a.ws[p] = (char*)d_ws_all[p];
  a.rank = rank;

  fused_kernel<<<NGEMM, 768, LDS_BYTES, stream>>>(a);
}
```
